```python
import numpy as np

kernel_name = "gpu_ag_gemm_m4096_k4096_n16384_w8a8_dqsilu"


ABSMAX_RTOL = 0.001

COMPUTE_DTYPE = "int8"
EPILOGUE = "dequant_silu"
IN_QUANT = "int8"
OUT_QUANT = None
IN_DTYPES = {"x": "int8", "w_mat": "int8",
             "scale_x": "float32", "scale_w": "float32"}
WORLD_SIZE = 8

def setup_inputs():

    x = np.random.randint(-127, 128, size=(4096, 4096)).astype(np.int8)
    w_mat = np.random.randint(-127, 128, size=(4096, 16384)).astype(np.int8)
    scale_x = (0.02 * (1.0 + np.random.rand(1))).astype(np.float32)
    scale_w = (0.02 * (1.0 + np.random.rand(1))).astype(np.float32)
    return {"x": x, "w_mat": w_mat, "scale_x": scale_x, "scale_w": scale_w}

def reference(x, w_mat, scale_x, scale_w):
    acc = x.astype(np.float64) @ w_mat.astype(np.float64)
    y = (acc.astype(np.float32) * (scale_x * scale_w)).astype(np.float32)
    return (y / (1.0 + np.exp(-np.clip(y, -60.0, 60.0)))).astype(np.float32)
```

```cpp
#include <hip/hip_runtime.h>
#include <rccl/rccl.h>
#include <cstdio>
#include <vector>
#include <algorithm>
#include <utility>

typedef unsigned u32x4 __attribute__((ext_vector_type(4)));
typedef int i32x4 __attribute__((ext_vector_type(4)));
typedef int i32x2 __attribute__((ext_vector_type(2)));
typedef int i32x16 __attribute__((ext_vector_type(16)));
typedef float f32x4 __attribute__((ext_vector_type(4)));
#define LAS __attribute__((address_space(3)))
#define OAUX 2
#define PSLEEP 2
#define DEEP 2
#define PAHEAD 2
#define HK 0
#define HL 1
#define KPI 27
#define PLINES 1

constexpr int NR = 8;
constexpr int MG = 4096, KG = 4096, NL = 2048, ML = 512;
constexpr unsigned SHARD = ML * KG;
constexpr int NWG = 256, NTHR = 256;
constexpr int NPUSH = 32;
constexpr size_t OFF_CNT = 0;
constexpr size_t OFF_TMO = 16384;
constexpr size_t OFF_FLAG = 32768;
constexpr unsigned FL_SRC = 36 * 128;
constexpr size_t OFF_STAMP = 131072;
constexpr size_t OFF_SLOT = 1u << 20;
constexpr unsigned long long DEADLINE = 50000000ull;
constexpr int LDS_DEAD = 131072, LDS_TOTAL = LDS_DEAD + 16;

struct Args { char* peer[NR - 1]; char* me; const char* x; const char* w; const float* sx; const float* sw; float* out; int rank; int var; int pace; int pace0; };

__device__ __forceinline__ unsigned rfl(unsigned v) { return __builtin_amdgcn_readfirstlane(v); }
#define WAITV(n) asm volatile("s_waitcnt vmcnt(" #n ")" ::: "memory")
#define WAITL0() asm volatile("s_waitcnt lgkmcnt(0)" ::: "memory")
#define BAR() do { asm volatile("" ::: "memory"); __builtin_amdgcn_s_barrier(); asm volatile("" ::: "memory"); } while (0)
#define SB() __builtin_amdgcn_sched_barrier(0)
#define PINS(x) asm volatile("" : "+s"(x))
#define PINV(x) asm volatile("" : "+v"(x))
template <int V> using IC = std::integral_constant<int, V>;
template <class F, int... I> __device__ __forceinline__ void sfor_impl(F&& f, std::integer_sequence<int, I...>) { (f(IC<I>{}), ...); }
template <int N, class F> __device__ __forceinline__ void sfor(F&& f) { sfor_impl(f, std::make_integer_sequence<int, N>{}); }

__device__ __forceinline__ void dma16(unsigned m0v, unsigned voff, __amdgpu_buffer_rsrc_t rs, unsigned soff) {
  asm volatile("s_mov_b32 m0, %0\n\ts_nop 0\n\tbuffer_load_dwordx4 %1, %2, %3 offen lds" :: "s"(m0v), "v"(voff), "s"(rs), "s"(soff) : "memory");
}
template <int OFF>
__device__ __forceinline__ void xload(const i32x4& d, unsigned voff, __amdgpu_buffer_rsrc_t rs, unsigned soff) {
  asm volatile("buffer_load_dwordx4 %0, %1, %2, %3 offen offset:%4 sc0 sc1" :: "v"(d), "v"(voff), "s"(rs), "s"(soff), "i"(OFF) : "memory");
}
__device__ __forceinline__ void pload(const unsigned& d, unsigned voff, __amdgpu_buffer_rsrc_t rs, unsigned soff) {
  asm volatile("buffer_load_dword %0, %1, %2, %3 offen sc0 sc1" :: "v"(d), "v"(voff), "s"(rs), "s"(soff) : "memory");
}
__device__ __forceinline__ float silu1(int acc, float sy, float st) {
  const float f = (float)acc;
  const float t = __builtin_amdgcn_fmed3f(f * st, -86.5617024533378f, 86.5617024533378f);
  return (f * sy) * __builtin_amdgcn_rcpf(1.0f + __builtin_amdgcn_exp2f(t));
}

__device__ __forceinline__ unsigned gemm_role(const Args& a, char* smem, int cg, int sidx, int src, int nloc, int loc0, int rot) {
  const int tid = threadIdx.x, lane = tid & 63;
  const int w = rfl(tid >> 6);
  const int nb = 32 + nloc;
  const unsigned ldsb = (unsigned)(size_t)(LAS char*)smem;
  const int g4 = lane >> 4, l15 = lane & 15;
  int rank = a.rank; PINS(rank); PINS(src);
  const __amdgpu_buffer_rsrc_t rsW = __builtin_amdgcn_make_buffer_rsrc((void*)a.w, 0, KG * NL, 0x00020000);
  const unsigned vw0 = (lane >> 2) * NL + (((lane & 3) ^ ((lane >> 4) & 1)) * 16), vw1 = vw0 ^ 32;
  const unsigned swb = (w * 1024) * NL + cg * 64;
  const unsigned m0b = ldsb + w * 32768;
  auto dma_gran = [&](int g) {
#pragma unroll
    for (int i = 0; i < 8; ++i) dma16(m0b + (g & 3) * 8192 + i * 1024, (i & 1) ? vw1 : vw0, rsW, swb + (((g + 2 * rot) & 7) * 128 + i * 16) * NL);
  };
  volatile LAS unsigned* dead = (volatile LAS unsigned*)(smem + LDS_DEAD);
  unsigned callv; u32x4 rt[3]; unsigned rdst[3];
  {
    const __amdgpu_buffer_rsrc_t rc = __builtin_amdgcn_make_buffer_rsrc(a.me + OFF_CNT, 0, NWG * 64, 0x00020000);
    asm volatile("buffer_load_dword %0, off, %1, %2 sc1" : "=v"(callv) : "s"(rc), "s"(blockIdx.x * 64) : "memory");
    const unsigned vl = l15 * KG + g4 * 16;
#pragma unroll
    for (int c = 0; c < 3; ++c) {
      const int F = (cg * 4 + w) * (nloc >> 1) + c, unit = loc0 + (F >> 6), f = F & 63;
      const __amdgpu_buffer_rsrc_t rx = __builtin_amdgcn_make_buffer_rsrc((void*)a.x, 0, c * 2 < nloc ? SHARD : 0u, 0x00020000);
      asm volatile("buffer_load_dwordx4 %0, %1, %2, %3 offen" : "=v"(rt[c]) : "v"(vl), "s"(rx), "s"(unit * (16 * KG) + f * 64) : "memory");
      rdst[c] = unit * 65536 + f * 1024;
    }
  }
  dma_gran(0);
  dead[0] = 0;
  asm volatile("s_waitcnt vmcnt(8)" : "+v"(callv), "+v"(rt[0]), "+v"(rt[1]), "+v"(rt[2]) :: "memory");
  const unsigned call = rfl(callv) + 1;
  const char* slot = a.me + OFF_SLOT + (size_t)(call & 1) * ((size_t)NR * SHARD);
#pragma unroll
  for (int c = 0; c < 3; ++c) {
    const __amdgpu_buffer_rsrc_t rd = __builtin_amdgcn_make_buffer_rsrc((void*)(slot + (size_t)rank * SHARD), 0, c * 2 < nloc ? SHARD : 0u, 0x00020000);
    __builtin_amdgcn_raw_buffer_store_b128(rt[c], rd, lane * 16 + rdst[c], 0, 17);
  }
  SB();
  dma_gran(1); dma_gran(2); dma_gran(3);
  float scale = a.sx[0] * a.sw[0];
  const int qq = l15 >> 1, pp = l15 & 1;
  const unsigned aw0 = w * 32768 + (g4 * 16 + qq) * 64 + pp * 8 + (((qq >> 2) | ((g4 & 1) << 1)) * 16);
  i32x4 wf[16][4];
#pragma unroll
  for (int g = 0; g < 8; ++g) {
    if (g <= 4) WAITV(24); else if (g == 5) WAITV(16); else if (g == 6) WAITV(8); else WAITV(0);
    if (g == 0) {
      WAITL0(); BAR();
      if (tid == 0) {
        __hip_atomic_store((unsigned*)(a.me + OFF_FLAG + (size_t)rank * FL_SRC) + sidx * 32 + cg, call, __ATOMIC_RELAXED, __HIP_MEMORY_SCOPE_SYSTEM);
        __hip_atomic_store((unsigned*)(a.me + OFF_CNT + blockIdx.x * 64), call, __ATOMIC_RELAXED, __HIP_MEMORY_SCOPE_AGENT);
      }
    }
#pragma unroll
    for (int kfl = 0; kfl < 2; ++kfl)
#pragma unroll
      for (int t = 0; t < 4; ++t) {
        const LAS char* p = (const LAS char*)smem + (aw0 ^ (t * 16)) + (g & 3) * 8192 + kfl * 4096;
        i32x2 lo = __builtin_amdgcn_ds_read_tr8_b64_v2i32((LAS i32x2*)p);
        i32x2 hi = __builtin_amdgcn_ds_read_tr8_b64_v2i32((LAS i32x2*)(p + 512));
        wf[g * 2 + kfl][t] = (i32x4){lo[0], lo[1], hi[0], hi[1]};
      }
    WAITL0();
#pragma unroll
    for (int kfl = 0; kfl < 2; ++kfl)
#pragma unroll
      for (int t = 0; t < 4; ++t) { if (g > 0) asm volatile("" : "+a"(wf[g * 2 + kfl][t])); else asm volatile("" : "+v"(wf[g * 2 + kfl][t])); }
    SB();
    if (g + 4 < 8) dma_gran(g + 4);
    SB();
  }
  PINV(scale);
  const float st = scale * -1.44269504088896f;
  const __amdgpu_buffer_rsrc_t rsF = __builtin_amdgcn_make_buffer_rsrc(a.me + OFF_FLAG, 0, NR * FL_SRC, 0x00020000);
  const unsigned vf = l15 * 4 + (PLINES == 2 ? (g4 & 1) * 128 : 0);
  const int fso = src * FL_SRC - nloc * 128;
  unsigned* tmo = (unsigned*)(a.me + OFF_TMO);
  const unsigned vst = lane * 16 + w * 16384;
  const unsigned vo = ((w * 4 + g4) * NL + cg * 64 + l15 * 4) * 4;
  const unsigned redr = l15 * 256 + (((w * 4 + g4) ^ l15) * 16), redr2 = redr + 65536;
  unsigned redw[4];
#pragma unroll
  for (int t = 0; t < 4; ++t) redw[t] = w * 32768 + t * 1024 + g4 * 256 + ((l15 ^ (t * 4 + g4)) * 16);
  unsigned ko[4];
#pragma unroll
  for (int q = 0; q < 4; ++q) ko[q] = ((q + rot) & 3) * 4096;
  int gsrc = src * 32; PINS(gsrc);
  auto nextg = [&](int g, int pos) -> int { return pos == nloc ? gsrc : g + 1; };
  int R = (a.var & 32) ? nb : nloc;
  auto absorb = [&](unsigned fv, int base) {
    unsigned lo = (unsigned)__ballot(fv == call); PINS(lo);
    const int r2 = base + (int)(PLINES == 2 ? (__builtin_ctzll(~(unsigned long long)lo) >> 4) : (__builtin_ctzll(~(unsigned long long)lo) >> 5));
    R = r2 > R ? r2 : R; PINS(R);
  };
  auto give_up = [&](unsigned long long t0) -> bool {
    const unsigned long long dt = __builtin_amdgcn_s_memrealtime() - t0;
    if (dt > 100000ull && (rfl(__hip_atomic_load(tmo, __ATOMIC_RELAXED, __HIP_MEMORY_SCOPE_AGENT)) == call || dt > DEADLINE)) {
      __hip_atomic_store(tmo, call, __ATOMIC_RELAXED, __HIP_MEMORY_SCOPE_AGENT);
      dead[0] = 1; return true;
    }
    return false;
  };
  auto wait_ready = [&](int n) {
    const unsigned long long t0 = __builtin_amdgcn_s_memrealtime();
    while (R <= n) {
      absorb(__builtin_amdgcn_raw_buffer_load_b32(rsF, vf, fso + R * 128, 17), R);
      asm volatile("" ::: "memory");
      if (R > n) break;
      if (give_up(t0)) R = nb;
      __builtin_amdgcn_s_sleep(PSLEEP);
    }
  };
  const __amdgpu_buffer_rsrc_t rsS = __builtin_amdgcn_make_buffer_rsrc((void*)slot, 0, NR * SHARD, 0x00020000);
  auto burst8 = [&](i32x4 (&xs)[16], int g, auto hc) __attribute__((always_inline)) {
    sfor<8>([&](auto jc) __attribute__((always_inline)) { constexpr int j = decltype(hc)::value * 8 + decltype(jc)::value; xload<(j & 3) * 1024>(xs[j], vst, rsS, g * 65536 + ko[j >> 2]); });
  };
  i32x4 xA[16], xB[16], acA[4], acB[4];
#pragma unroll
  for (int t = 0; t < 4; ++t) { acA[t] = (i32x4){0, 0, 0, 0}; acB[t] = (i32x4){0, 0, 0, 0}; }
#pragma unroll
  for (int j = 0; j < 16; ++j) { asm volatile("" : "=v"(xA[j])); asm volatile("" : "=v"(xB[j])); }
  unsigned fvB = 0; PINV(fvB);
  {
    const unsigned long long t0 = __builtin_amdgcn_s_memrealtime();
    while (!(a.var & 32)) {
      const unsigned fv = __builtin_amdgcn_raw_buffer_load_b32(rsF, (lane & 31) * 4, rank * FL_SRC + sidx * 128, 17);
      asm volatile("" ::: "memory");
      if (__ballot(fv == call) == ~0ull) break;
      if (give_up(t0)) { R = nb; break; }
      __builtin_amdgcn_s_sleep(PSLEEP);
    }
  }
  int gout = 0, g0 = rank * 32 + loc0, g1 = g0 + 1;
  burst8(xA, g0, IC<0>{}); burst8(xA, g0, IC<1>{});
  unsigned orec = 0;
  int i = 0;
  auto stream = [&](i32x4 (&xc)[16], i32x4 (&xn)[16], i32x4 (&ac)[4], i32x4 (&pv)[4], auto hbc, auto mc) __attribute__((always_inline)) {
    constexpr int hb = decltype(hbc)::value;
    constexpr bool FQ = decltype(mc)::value != 0;
    const __amdgpu_buffer_rsrc_t rsO = __builtin_amdgcn_make_buffer_rsrc((void*)a.out, 0, orec, 0x00020000);
    const __amdgpu_buffer_rsrc_t rsN = __builtin_amdgcn_make_buffer_rsrc((void*)slot, 0, (FQ || hb == 0 || i + 1 < nb) ? NR * SHARD : 0u, 0x00020000);
    const unsigned sn = g1 * 65536;
    i32x4 p0, p1, p2, p3; float f[4], t[4], sy; unsigned dd, va, lo; int gn, baseB;
    const int plim = i + 2 < nb - 1 ? i + 2 : nb - 1;
    sfor<64>([&](auto kc) __attribute__((always_inline)) {
      constexpr int k = decltype(kc)::value, j = k >> 2, tl = k & 3;
      if (tl == 0) {
        constexpr int cnt = FQ ? 15 : (j > HK ? 15 - j + ((j - HK) * HL < 16 ? (j - HK) * HL : 16) : 15 - j);
        asm volatile("s_waitcnt vmcnt(%1)" : "+v"(xc[j]) : "i"(cnt) : "memory");
      }
      ac[tl] = __builtin_amdgcn_mfma_i32_16x16x64_i8(wf[j][tl], xc[j], j == 0 ? (i32x4){0, 0, 0, 0} : ac[tl], 0, 0, 0);
      SB();
      if (FQ && tl == 0) xload<(j & 3) * 1024>(xn[j], vst, rsS, sn + ko[j >> 2]);
      if (!FQ && k == HK * 4) {
        if (R <= i + 1 && i + 1 < nb) {
          wait_ready(i + 1);
        }
      }
      if constexpr (!FQ && j >= HK && (tl * HL) % 4 == 0) {
        constexpr int m = (j - HK) * HL + tl * HL / 4;
        if constexpr (m < 16) xload<(m & 3) * 1024>(xn[m], vst, rsN, sn + ko[m >> 2]);
      }
      if (k == (FQ ? 3 : KPI)) {
        const __amdgpu_buffer_rsrc_t rsP = __builtin_amdgcn_make_buffer_rsrc(a.me + OFF_FLAG, 0, R <= (FQ ? i + 4 - hb : plim) ? NR * FL_SRC : 0u, 0x00020000);
        baseB = R; pload(fvB, vf, rsP, fso + R * 128);
      }
      constexpr int lH_after = (HK + (16 + HL - 1) / HL <= 15 ? 16 : (15 - HK) * HL) - ((KPI / 4 + 1 > HK) ? ((KPI / 4 + 1 - HK) * HL < 16 ? (KPI / 4 + 1 - HK) * HL : 16) : 0);
      if (k == 59) { asm volatile("s_waitcnt vmcnt(%1)" : "+v"(fvB) : "i"(FQ ? 14 : lH_after) : "memory"); lo = (unsigned)__ballot(fvB == call); PINS(lo); }
      if (k == 61) { lo = (unsigned)(PLINES == 2 ? (__builtin_ctzll(~(unsigned long long)lo) >> 4) : (__builtin_ctzll(~(unsigned long long)lo) >> 5)); PINS(lo); }
      if (k == 62) { const int r2 = baseB + (int)lo; R = r2 > R ? r2 : R; PINS(R); }
      if (k == 11) { gn = nextg(g1, i + 2); PINS(gn); }
      if (k == 1 || k == 5 || k == 9 || k == 13) *(LAS i32x4*)(smem + redw[(k >> 2) & 3] + (hb ^ 1) * 4096) = pv[(k >> 2) & 3];
      if (k == 17) { WAITL0(); if (__builtin_expect(orec != 0, 1)) BAR(); }
      if (k == 18) { p0 = *(const LAS i32x4*)(smem + redr + (hb ^ 1) * 4096); p1 = *(const LAS i32x4*)(smem + redr + (hb ^ 1) * 4096 + 32768); }
      if (k == 19) { p2 = *(const LAS i32x4*)(smem + redr2 + (hb ^ 1) * 4096); p3 = *(const LAS i32x4*)(smem + redr2 + (hb ^ 1) * 4096 + 32768); }
      if (k == 21) { dd = dead[0]; va = vo + gout * (16 * NL * 4); PINV(va); }
      constexpr int e = tl == 0 ? -1 : (j - 7) * 3 + tl - 1;
      if (e == 0 || e == 1) { _Pragma("unroll") for (int c = 0; c < 2; ++c) { p0[(e & 1) * 2 + c] += p1[(e & 1) * 2 + c]; } PINV(p0); }
      if (e == 2 || e == 3) { _Pragma("unroll") for (int c = 0; c < 2; ++c) { p0[(e & 1) * 2 + c] += p2[(e & 1) * 2 + c] + p3[(e & 1) * 2 + c]; } PINV(p0); }
      if (e == 4 || e == 5) { _Pragma("unroll") for (int c = (e & 1) * 2; c < (e & 1) * 2 + 2; ++c) { f[c] = (float)p0[c]; PINV(f[c]); } }
      if (e == 6 || e == 7) { _Pragma("unroll") for (int c = (e & 1) * 2; c < (e & 1) * 2 + 2; ++c) { t[c] = f[c] * st; PINV(t[c]); } }
      if (e == 8 || e == 9) { _Pragma("unroll") for (int c = (e & 1) * 2; c < (e & 1) * 2 + 2; ++c) { t[c] = __builtin_amdgcn_fmed3f(t[c], -86.5617024533378f, 86.5617024533378f); PINV(t[c]); } }
      if (e >= 10 && e < 14) { t[(e - 10) & 3] = __builtin_amdgcn_exp2f(t[(e - 10) & 3]); PINV(t[(e - 10) & 3]); }
      if (k == 44) { sy = dd ? __builtin_nanf("") : scale; PINV(sy); }
      if (e == 14 || e == 15) { _Pragma("unroll") for (int c = (e & 1) * 2; c < (e & 1) * 2 + 2; ++c) { t[c] = 1.0f + t[c]; PINV(t[c]); } }
      if (e >= 16 && e < 20) { t[(e - 16) & 3] = __builtin_amdgcn_rcpf(t[(e - 16) & 3]); PINV(t[(e - 16) & 3]); }
      if (k == 48) { f[0] *= sy; PINV(f[0]); }
      if (k == 52) { f[1] *= sy; PINV(f[1]); }
      if (e == 20) { f[2] *= sy; PINV(f[2]); f[3] *= sy; PINV(f[3]); }
      if (e == 21 || e == 22) { _Pragma("unroll") for (int c = ((e + 1) & 1) * 2; c < ((e + 1) & 1) * 2 + 2; ++c) { f[c] = f[c] * t[c]; PINV(f[c]); } }
      if (e == 24) __builtin_amdgcn_raw_buffer_store_b128(__builtin_bit_cast(u32x4, (f32x4){f[0], f[1], f[2], f[3]}), rsO, va, 0, OAUX);
      SB();
    });
    orec = MG * NL * 4; gout = g0;
    g0 = g1; g1 = gn; ++i;
  };
  do {
    if (R > i + 2) {
      do { stream(xA, xB, acA, acB, IC<0>{}, IC<1>{}); stream(xB, xA, acB, acA, IC<1>{}, IC<1>{}); } while (R > i + 2);
    }
    stream(xA, xB, acA, acB, IC<0>{}, IC<0>{}); stream(xB, xA, acB, acA, IC<1>{}, IC<0>{});
  } while (i < nb);
  {
#pragma unroll
    for (int q = 0; q < 4; ++q) *(LAS i32x4*)(smem + redw[q] + 4096) = acB[q];
    WAITL0(); BAR();
    const i32x4 q0 = *(const LAS i32x4*)(smem + redr + 4096), q1 = *(const LAS i32x4*)(smem + redr + 4096 + 32768), q2 = *(const LAS i32x4*)(smem + redr2 + 4096), q3 = *(const LAS i32x4*)(smem + redr2 + 4096 + 32768);
    const float sy2 = dead[0] ? __builtin_nanf("") : scale;
    const i32x4 k2 = q3 + q0 + q1 + q2;
    f32x4 o2;
#pragma unroll
    for (int j = 0; j < 4; ++j) o2[j] = silu1(k2[j], sy2, st);
    const __amdgpu_buffer_rsrc_t rsO2 = __builtin_amdgcn_make_buffer_rsrc((void*)a.out, 0, MG * NL * 4, 0x00020000);
    __builtin_amdgcn_raw_buffer_store_b128(__builtin_bit_cast(u32x4, o2), rsO2, vo + gout * (16 * NL * 4), 0, OAUX);
#pragma unroll
    for (int j = 0; j < 16; ++j) { asm volatile("" :: "v"(xA[j])); asm volatile("" :: "v"(xB[j])); }
  }
  return call;
}

__device__ __forceinline__ void push_role(const Args& a, unsigned callv, int p) {
  const int lane = threadIdx.x & 63;
  const int par = p >> 4;
  const __amdgpu_buffer_rsrc_t rx = __builtin_amdgcn_make_buffer_rsrc((void*)a.x, 0, SHARD, 0x00020000);
  const unsigned vl = (lane & 15) * KG + (lane >> 4) * 16 + par * (16 * KG), vs = lane * 16 + par * 65536;
  auto srco = [&](int r) -> unsigned { return r * (32 * KG) + ((p + r) & 15) * 256; };
  u32x4 v[2][4];
#pragma unroll
  for (int j = 0; j < 4; ++j) { v[0][j] = __builtin_amdgcn_raw_buffer_load_b128(rx, vl + j * 64, srco(0), 0); SB(); }
  char* pl = a.peer[0];
#pragma unroll
  for (int i = 1; i < 7; ++i) pl = lane == i ? a.peer[i] : pl;
  unsigned* flag = (unsigned*)(pl + OFF_FLAG + (size_t)a.rank * FL_SRC) + (p & 15) + par * 32;
  const unsigned call = rfl(callv) + 1;
  const size_t so = OFF_SLOT + (size_t)(call & 1) * ((size_t)NR * SHARD) + (size_t)a.rank * SHARD;
  __amdgpu_buffer_rsrc_t rs[7];
#pragma unroll
  for (int i = 0; i < 7; ++i) rs[i] = __builtin_amdgcn_make_buffer_rsrc(a.peer[i] + so, 0, SHARD, 0x00020000);
  const unsigned per = a.pace;
  const unsigned tb = (unsigned)__builtin_amdgcn_s_memrealtime();
  for (int it = 0; it < 8; ++it) {
#pragma unroll
    for (int h = 0; h < 2; ++h) {
      const int r = it * 2 + h;
#pragma unroll
      for (int j = 0; j < 4; ++j) {
        const int pc = (2 * r + par) * 4 + j - a.pace0;
        if (pc > 0) { const unsigned due = (pc * per) / 40u; while ((unsigned)__builtin_amdgcn_s_memrealtime() - tb < due) __builtin_amdgcn_s_sleep(1); }
        SB();
#pragma unroll
        for (int i = 0; i < 7; ++i) __builtin_amdgcn_raw_buffer_store_b128(v[h][j], rs[i], vs + j * 1024, r * 131072 + ((p + r) & 15) * 4096, 17);
        SB();
        if (j == 0) {
#pragma unroll
          for (int jj = 0; jj < 4; ++jj) v[h ^ 1][jj] = __builtin_amdgcn_raw_buffer_load_b128(rx, vl + jj * 64, srco((r + 1) & 15), 0);
          SB();
        }
      }
      WAITV(0);
      if (lane < 7) __hip_atomic_store(flag + r * 64, call, __ATOMIC_RELAXED, __HIP_MEMORY_SCOPE_SYSTEM);
      SB();
    }
  }
}

__global__ void __launch_bounds__(NTHR, 1) fused_ag_gemm(Args a) {
  __shared__ __attribute__((aligned(1024))) char smem[LDS_TOTAL];
  const int b = blockIdx.x;
  const int wv = rfl(threadIdx.x >> 6);
  unsigned call = 0;
  if (b < NPUSH) {
    if (wv == 0) {
      unsigned* cnt = (unsigned*)(a.me + OFF_CNT + b * 64);
      const unsigned callv = __hip_atomic_load(cnt, __ATOMIC_RELAXED, __HIP_MEMORY_SCOPE_AGENT);
      if (!(a.var & 32)) push_role(a, callv, b);
      call = callv + 1;
      if (threadIdx.x == 0) __hip_atomic_store(cnt, call, __ATOMIC_RELAXED, __HIP_MEMORY_SCOPE_AGENT);
    }
  } else {
    const int lg = (a.var >> 8) & 7, G = 4 << lg;
    const int q = (b & 7) * 28 + (b >> 3) - 4; const int grp = q / (7 * G), r = q % (7 * G), s = r / G;
    call = gemm_role(a, smem, grp * G + (r % G), s, (a.rank + 1 + s) & 7, s < 2 ? 6 : 4, s < 2 ? 6 * s : 4 + 4 * s, (a.var & 128) ? (s & 3) : 0);
  }
}

static int g_var = 128 + 256 + (1150 << 12) + (2 << 1);
extern "C" void kernel_launch(int rank, int nranks, ncclComm_t comm, hipStream_t stream, void* const* d_in, const int* in_sizes, int n_in,
                              void* d_out, int out_size, const int* in_dtypes, int out_dtype, void* const* d_out_all, void* const* d_ws_all, size_t ws_bytes) {
  if (nranks != NR || (unsigned)rank >= (unsigned)NR || ws_bytes < OFF_SLOT + 2ull * NR * SHARD) return;
  Args a; for (int i = 0; i < NR - 1; ++i) a.peer[i] = (char*)d_ws_all[(rank + 1 + i) & 7];
  a.me = (char*)d_ws_all[rank];
  a.x = (const char*)d_in[0]; a.w = (const char*)d_in[1]; a.sx = (const float*)d_in[2]; a.sw = (const float*)d_in[3];
  a.out = (float*)d_out; a.rank = rank; a.var = g_var;
  a.pace = (g_var >> 12) & 0xfff; a.pace0 = (g_var >> 1) & 7;
  fused_ag_gemm<<<NWG, NTHR, 0, stream>>>(a);
}
```
